```python
import numpy as np

kernel_name = "gpu_a2a_gemm_m4096_k4096_n4096_f32_silu"


ABSMAX_RTOL = 0.001

COMPUTE_DTYPE = "float32"
EPILOGUE = "silu"
IN_QUANT = None
OUT_QUANT = None
IN_DTYPES = {"x": "float32", "w_mat": "float32"}
WORLD_SIZE = 8

def setup_inputs():

    x = np.random.randn(4096, 4096).astype(np.float32)
    w_mat = (np.random.randn(4096, 4096) / np.sqrt(4096)).astype(np.float32)
    return {"x": x, "w_mat": w_mat}

def _epilogue(y):
    return (y / (1.0 + np.exp(-np.clip(y, -60.0, 60.0)))).astype(np.float32)

def reference(x, w_mat):
    y = (x @ w_mat).astype(np.float32)
    return _epilogue(y)
```

```cpp
#include <hip/hip_runtime.h>
#include <rccl/rccl.h>
#include <cstdio>

#define HIP_CHECK(expr) do { hipError_t _e = (expr); if (_e != hipSuccess) { \
    fprintf(stderr, "HIP error %s at %s:%d\n", hipGetErrorString(_e), __FILE__, __LINE__); \
    return; } } while (0)

namespace {

typedef unsigned v4u __attribute__((ext_vector_type(4)));
typedef float f32x4 __attribute__((ext_vector_type(4)));
typedef _Float16 h8 __attribute__((ext_vector_type(8)));
typedef _Float16 h2 __attribute__((ext_vector_type(2)));
#define LAS __attribute__((address_space(3)))
#define GAS __attribute__((address_space(1)))

constexpr int NR = 8;
constexpr int MR = 512;
constexpr int KD = 4096, ND = 4096;
constexpr int KR = 512;
constexpr int NWG = 256;
constexpr int NB = 2;
constexpr int NWAVES = 4 + NB + 2;
constexpr int NKS = 64;
#define UDEPTH 2
constexpr int UD = UDEPTH;
constexpr int RU = 32;
constexpr int RING_BYTES = RU * 4096;
constexpr int LDS_BYTES = RING_BYTES + 512 + 4096;

#define POLL_SCALAR 0
#define XLD_EARLY 16
#define XLD_LEAD 200
#define LD_DELAY 0
#define PRIO 3
#define SEND_T0 100
#define SEND_DT 80
constexpr unsigned WS_EPOCH = 0;
constexpr unsigned WS_TMO = 64u << 10;
constexpr unsigned WS_RFLAG = 128u << 10;
constexpr unsigned WS_PROG = 256u << 10;
constexpr unsigned WS_STAGE = 1u << 20;
constexpr unsigned STAGE_BYTES = 4u << 20;
constexpr unsigned WS_SLAB = 16u << 20;
constexpr unsigned SLAB_BYTES = 8u << 20;
constexpr unsigned WS_USED = WS_SLAB + SLAB_BYTES;
constexpr unsigned DEADLINE_TICKS = 80000000u;
constexpr unsigned DEADLINE_LDS = 190000000u;

#define POLL_SLEEP 4
constexpr int AUX_SYS = 17;
constexpr int AUX_DEV = 16;
constexpr int RSRC_W3 = 0x00020000;

#define STAMP(k) do {} while (0)
#define STAMPV(k, v) do {} while (0)

struct Args {
  const float* x;
  const float* w;
  float* out;
  unsigned char* ws[NR];
  int rank;
};

__device__ __forceinline__ unsigned cvt2(float a, float b) {
  h2 r; r[0] = (_Float16)a; r[1] = (_Float16)b; return __builtin_bit_cast(unsigned, r);
}
__device__ __forceinline__ f32x4 mfma(v4u a, v4u b, f32x4 c) {
  return __builtin_amdgcn_mfma_f32_16x16x32_f16(__builtin_bit_cast(h8, a), __builtin_bit_cast(h8, b), c, 0, 0, 0);
}
#define SCHED() __builtin_amdgcn_sched_barrier(0)
#define WAIT_L0() asm volatile("s_waitcnt lgkmcnt(0)" ::: "memory")
#define WAIT_V0() asm volatile("s_waitcnt vmcnt(0)" ::: "memory")

__device__ __forceinline__ int kseq(int i, int q, int rank) {
  int d, ff;
  if (i < 8) { d = 0; ff = i; }
  else { int j = i - 8; ff = j / 7; d = 1 + j % 7; }
  return ((((rank + d) & 7) << 4) + 2 * ff + q) & 127;
}

__global__ void __launch_bounds__(NWAVES * 64, 2) a2a_gemm_silu(Args p) {
  __shared__ __attribute__((aligned(16))) unsigned char smem[LDS_BYTES];
  LAS unsigned char* L = (LAS unsigned char*)smem;
  volatile LAS unsigned* ctl = (volatile LAS unsigned*)(L + RING_BYTES);
  const int tid = threadIdx.x, lane = tid & 63, w = __builtin_amdgcn_readfirstlane(tid >> 6);
  const int b = blockIdx.x, xcd = b & 7, idx = b >> 3;
  unsigned char* const wsl = p.ws[p.rank];
  const unsigned t0 = (unsigned)__builtin_amdgcn_s_memrealtime();

  if (w == 4 + NB) {
    const int dd = b >> 5, u = (b & 24) | ((b + dd) & 7);
    const int dest = (p.rank + dd) & 7, f = u >> 1, mts = u & 1;
    __builtin_amdgcn_s_setprio(PRIO);
    const unsigned tl = dd == 0 ? 0u : SEND_T0 + f * SEND_DT;
    const bool early = dd == 0 || f < XLD_EARLY;
    if (!early) {
      __builtin_amdgcn_s_barrier();
      while ((unsigned)__builtin_amdgcn_s_memrealtime() - t0 + XLD_LEAD < tl) __builtin_amdgcn_s_sleep(2);
    }
    GAS unsigned* e = (GAS unsigned*)(wsl + WS_EPOCH) + b * 32 + w;
    const unsigned ev = __hip_atomic_load(e, __ATOMIC_RELAXED, __HIP_MEMORY_SCOPE_AGENT);
    const __amdgpu_buffer_rsrc_t rsX = __builtin_amdgcn_make_buffer_rsrc((void*)p.x, 0, KD * KR * 4, RSRC_W3);
    const int voffX = (lane & 15) * (KR * 4) + (lane >> 4) * 32;
    const int soX = (dest * MR + mts * 256) * (KR * 4) + f * 128;
    f32x4 xr[16][2];
#pragma unroll
    for (int m = 0; m < 16; ++m) {
      xr[m][0] = __builtin_bit_cast(f32x4, __builtin_amdgcn_raw_buffer_load_b128(rsX, voffX, soX + m * (16 * KR * 4), 0));
      xr[m][1] = __builtin_bit_cast(f32x4, __builtin_amdgcn_raw_buffer_load_b128(rsX, voffX + 16, soX + m * (16 * KR * 4), 0));
    }
    SCHED();
    if (early) __builtin_amdgcn_s_barrier();
    SCHED();
    while ((unsigned)__builtin_amdgcn_s_memrealtime() - t0 < tl) __builtin_amdgcn_s_sleep(2);
    SCHED();
    const unsigned call = __builtin_amdgcn_readfirstlane(ev) + 1u;
    const __amdgpu_buffer_rsrc_t rsD = __builtin_amdgcn_make_buffer_rsrc((void*)(p.ws[dest] + WS_STAGE), 0, 2 * STAGE_BYTES, RSRC_W3);
    const int soD = (int)((call & 1u) * STAGE_BYTES) + ((p.rank * 16 + f) * 32 + mts * 16) * 1024;
#pragma unroll
    for (int m = 0; m < 16; ++m) {
      v4u o;
      o[0] = cvt2(xr[m][0][0], xr[m][0][1]); o[1] = cvt2(xr[m][0][2], xr[m][0][3]);
      o[2] = cvt2(xr[m][1][0], xr[m][1][1]); o[3] = cvt2(xr[m][1][2], xr[m][1][3]);
      __builtin_amdgcn_raw_buffer_store_b128(o, rsD, lane * 16 + (m & 3) * 1024, soD + (m >> 2) * 4096, AUX_SYS);
      SCHED();
    }
    WAIT_V0();
    if (lane == 0) {
      const int d = (8 - dd) & 7;
      GAS unsigned* fl = (GAS unsigned*)(p.ws[dest] + WS_PROG) + (mts * 2 + (f & 1)) * 64 + (d == 0 ? (f >> 1) : 8 + (f >> 1) * 7 + (d - 1));
      __hip_atomic_store(fl, call, __ATOMIC_RELAXED, __HIP_MEMORY_SCOPE_SYSTEM);
      __hip_atomic_store(e, call, __ATOMIC_RELAXED, __HIP_MEMORY_SCOPE_AGENT);
    }
    return;
  }

  const int q = xcd >> 2, mt = idx & 1, ntl = idx >> 1;
  const int n0 = (xcd & 3) * 1024 + ntl * 64;
  const int ktab = kseq(lane, q, p.rank);
#define KS(i) (__builtin_amdgcn_readlane(ktab, (i) & 63))
  const unsigned ctl_addr = RING_BYTES;
  if (tid < 16) ctl[tid] = 0;
  WAIT_L0();
  __builtin_amdgcn_s_barrier();

  if (w == 5 + NB) {
    GAS unsigned* e = (GAS unsigned*)(wsl + WS_EPOCH) + b * 32 + w;
    const unsigned call = __builtin_amdgcn_readfirstlane(__hip_atomic_load(e, __ATOMIC_RELAXED, __HIP_MEMORY_SCOPE_AGENT)) + 1u;
    __builtin_amdgcn_s_setprio(PRIO);
    GAS unsigned* const tmo = (GAS unsigned*)(wsl + WS_TMO);
    int pub = 0;
    for (unsigned it = 1;; ++it) {
      GAS unsigned* pw = (GAS unsigned*)(wsl + WS_PROG) + (mt * 2 + q) * 64 + lane;
      const unsigned v = __hip_atomic_load(pw, __ATOMIC_RELAXED, __HIP_MEMORY_SCOPE_SYSTEM);
      const unsigned long long missing = __builtin_amdgcn_ballot_w64((int)(v - call) < 0);
      const int rdy = missing ? __builtin_ctzll(missing) : NKS;
      if (rdy != pub) { pub = rdy; ctl[2] = rdy; }
      if (rdy >= NKS) break;
      if ((it & 63u) == 0u) {
        if ((unsigned)__builtin_amdgcn_s_memrealtime() - t0 > DEADLINE_TICKS ||
            __hip_atomic_load(tmo, __ATOMIC_RELAXED, __HIP_MEMORY_SCOPE_AGENT) == call) {
          if (lane == 0) __hip_atomic_store(tmo, call, __ATOMIC_RELAXED, __HIP_MEMORY_SCOPE_AGENT);
          ctl[3] = 1;
          WAIT_L0();
          ctl[2] = NKS;
          break;
        }
      }
      __builtin_amdgcn_s_sleep(POLL_SLEEP);
    }
    STAMP(0);
    if (lane == 0) __hip_atomic_store(e, call, __ATOMIC_RELAXED, __HIP_MEMORY_SCOPE_AGENT);
    return;
  }
  if (w >= 4) {
    const int j = w - 4;
    const int voffB = (lane >> 4) * 8 * (ND * 4) + (lane & 15) * 16 + n0 * 4;
    f32x4 br[UD][8];
#define LOADB(r, u) do { const int u_ = (u); const int so_ = KS(u_) * (32 * ND * 4);                                 \
      const __amdgpu_buffer_rsrc_t rs_ = __builtin_amdgcn_make_buffer_rsrc((void*)p.w, 0, u_ < NKS ? KD * ND * 4 : 0, RSRC_W3); \
      _Pragma("unroll") for (int jj = 0; jj < 8; ++jj)                                                              \
        br[r][jj] = __builtin_bit_cast(f32x4, __builtin_amdgcn_raw_buffer_load_b128(rs_, voffB, so_ + jj * (ND * 4), 0)); } while (0)
#pragma unroll
    for (int r = 0; r < UD; ++r) { LOADB(r, r * NB + j); SCHED(); }
    int freed = RU;
    for (int n_ = 0; n_ * NB + j < NKS; n_ += UD) {
#pragma unroll
      for (int r = 0; r < UD; ++r) {
        const int n = n_ + r, u = n * NB + j;
        v4u bf[4];
#pragma unroll
        for (int t = 0; t < 4; ++t)
#pragma unroll
          for (int jj = 0; jj < 4; ++jj) bf[t][jj] = cvt2(br[r][2 * jj][t], br[r][2 * jj + 1][t]);
        asm volatile("" : "+v"(bf[0]), "+v"(bf[1]), "+v"(bf[2]), "+v"(bf[3]));
        SCHED();
        LOADB(r, u + UD * NB);
        SCHED();
        {
          const int uw = u < NKS ? u : 0;
          unsigned t0_, t1_, t2_, t3_;
          asm volatile("s_cmp_lt_i32 %[uu], %[fr]\n\ts_cbranch_scc1 .Lfok%=\n"
                       ".Lfp%=:\n\tds_read_b32 %[t0], %[ad] offset:16\n\tds_read_b32 %[t1], %[ad] offset:20\n\t"
                       "ds_read_b32 %[t2], %[ad] offset:24\n\tds_read_b32 %[t3], %[ad] offset:28\n\t"
                       "s_memrealtime vcc\n\ts_waitcnt lgkmcnt(0)\n\t"
                       "v_min_u32 %[t0], %[t0], %[t1]\n\tv_min3_u32 %[t0], %[t0], %[t2], %[t3]\n\ts_nop 0\n\t"
                       "v_readfirstlane_b32 %[fr], %[t0]\n\ts_add_i32 %[fr], %[fr], %[ru]\n\t"
                       "s_cmp_lt_i32 %[uu], %[fr]\n\ts_cbranch_scc1 .Lfok%=\n\t"
                       "s_sleep 4\n\ts_sub_u32 vcc_lo, vcc_lo, %[ts]\n\ts_cmp_lt_u32 vcc_lo, %[dl]\n\ts_cbranch_scc1 .Lfp%=\n"
                       ".Lfok%=:"
                       : [fr] "+&s"(freed), [t0] "=&v"(t0_), [t1] "=&v"(t1_), [t2] "=&v"(t2_), [t3] "=&v"(t3_)
                       : [uu] "s"(uw), [ad] "v"(ctl_addr), [ru] "i"(RU), [ts] "s"(t0), [dl] "s"(DEADLINE_LDS)
                       : "scc", "vcc", "memory");
          freed = __builtin_amdgcn_readfirstlane(freed);
          LAS v4u* slot = (LAS v4u*)(L + (u < NKS ? ((unsigned)u % RU) * 4096 : RING_BYTES + 512));
#pragma unroll
          for (int t = 0; t < 4; ++t) slot[t * 64 + lane] = bf[t];
          WAIT_L0();
          ctl[j] = n + 1;
          SCHED();
        }
      }
    }
    return;
  }

  GAS unsigned* const epoch = (GAS unsigned*)(wsl + WS_EPOCH) + b * 32 + w;
  const unsigned epv = __hip_atomic_load(epoch, __ATOMIC_RELAXED, __HIP_MEMORY_SCOPE_AGENT);
  GAS unsigned* const tmo = (GAS unsigned*)(wsl + WS_TMO);
  const int voffA0 = lane * 16 + (mt * 16 + w * 4 + 2 * q) * 1024;
  const int voffA1 = lane * 16 + (mt * 16 + w * 4 + 2 * (q ^ 1)) * 1024;
  f32x4 acc[4][4];
#pragma unroll
  for (int m = 0; m < 4; ++m)
#pragma unroll
    for (int t = 0; t < 4; ++t) acc[m][t] = f32x4{0.f, 0.f, 0.f, 0.f};
  v4u a[2][4][4];
#define RSA(i) __builtin_amdgcn_make_buffer_rsrc((void*)stage, 0, (i) < NKS ? (int)STAGE_BYTES : 0, RSRC_W3)
#define LOADA1(h, kk, m, rs, so) a[h][kk][m] = __builtin_amdgcn_raw_buffer_load_b128(rs, ((m) < 2 ? voffA0 : voffA1) + ((m) & 1) * 1024, so, AUX_SYS)
  int readyb = 0, readya = 0;
  unsigned failed = 0;
#define WAITAB(ub, ua) do { unsigned t0_, t1_, t2_;                                                                  \
    asm volatile("s_cmp_lt_i32 %[u1], %[rb]\n\ts_cselect_b32 vcc_lo, %[ra], 0\n\t"                                   \
                 "s_cmp_lt_i32 %[u2], vcc_lo\n\ts_cbranch_scc1 .Lbok%=\n"                                            \
                 ".Lbp%=:\n\tds_read_b32 %[t0], %[ad]\n\tds_read_b32 %[t1], %[ad] offset:4\n\tds_read_b32 %[t2], %[ad] offset:8\n\t" \
                 "s_memrealtime vcc\n\ts_waitcnt lgkmcnt(0)\n\t"                                                     \
                 "v_lshlrev_b32 %[t0], 1, %[t0]\n\tv_lshl_or_b32 %[t1], %[t1], 1, 1\n\t"                            \
                 "v_min_u32 %[t0], %[t0], %[t1]\n\tv_readfirstlane_b32 %[ra], %[t2]\n\tv_readfirstlane_b32 %[rb], %[t0]\n\t" \
                 "s_sub_u32 vcc_hi, vcc_lo, %[ts]\n\t"                                                               \
                 "s_cmp_lt_i32 %[u1], %[rb]\n\ts_cselect_b32 vcc_lo, %[ra], 0\n\t"                                   \
                 "s_cmp_lt_i32 %[u2], vcc_lo\n\ts_cbranch_scc1 .Lbok%=\n\t"                                          \
                 "s_sleep 1\n\ts_cmp_lt_u32 vcc_hi, %[dl]\n\ts_cbranch_scc1 .Lbp%=\n\t"                              \
                 "s_mov_b32 %[fl], 1\n"                                                                              \
                 ".Lbok%=:"                                                                                          \
                 : [rb] "+&s"(readyb), [ra] "+&s"(readya), [fl] "+&s"(failed), [t0] "=&v"(t0_), [t1] "=&v"(t1_), [t2] "=&v"(t2_) \
                 : [u1] "s"(ub), [u2] "s"(ua), [ad] "v"(ctl_addr), [ts] "s"(t0), [dl] "s"(DEADLINE_LDS) : "scc", "vcc", "memory"); \
    readyb = __builtin_amdgcn_readfirstlane(readyb); readya = __builtin_amdgcn_readfirstlane(readya);                \
    failed = __builtin_amdgcn_readfirstlane(failed); } while (0)
#define LSTAMP(k) do {} while (0)
#define LSTAMPC(k) do {} while (0)
  v4u bq[2][4];
  const LAS v4u* lbase = (const LAS v4u*)L + lane;
#define READB1(buf, u, t) bq[buf][t] = lbase[(((unsigned)(u) % RU) * 4 + (t)) * 64]
  const unsigned call = __builtin_amdgcn_readfirstlane(epv) + 1u;
  unsigned char* const stage = wsl + WS_STAGE + (call & 1u) * STAGE_BYTES;
  const __amdgpu_buffer_rsrc_t rsA = RSA(0);
  __builtin_amdgcn_s_waitcnt(0xC07F);
#pragma unroll
  for (int h = 0; h < 2; ++h) {
    WAITAB(0, h * 4 + 3);
    if (h == 0) { LSTAMP(0); LSTAMPC(12); }
#pragma unroll
    for (int kk = 0; kk < 4; ++kk) {
      const int so = KS(h * 4 + kk) * 32768;
#pragma unroll
      for (int m = 0; m < 4; ++m) LOADA1(h, kk, m, rsA, so);
      SCHED();
    }
  }
#pragma unroll
  for (int t = 0; t < 4; ++t) READB1(0, 0, t);
  for (int s0 = 0; s0 < NKS; s0 += 8) {
    LSTAMP(1 + (s0 >> 3));
#pragma unroll
    for (int h = 0; h < 2; ++h) {
      const int i0 = s0 + h * 4;
      WAITAB(min(i0 + 4, NKS - 1), min(i0 + 11, NKS - 1));
      const __amdgpu_buffer_rsrc_t rs = RSA(i0 + 8);
      SCHED();
#pragma unroll
      for (int kk = 0; kk < 4; ++kk) {
        const int cur = kk & 1;
        const int so = KS(i0 + kk + 8) * 32768;
        const int un = i0 + kk + 1;
#pragma unroll
        for (int m = 0; m < 4; ++m) {
#pragma unroll
          for (int t = 0; t < 4; ++t) {
            acc[m][t] = mfma(a[h][kk][m], bq[cur][t], acc[m][t]);
            asm volatile("" : "+v"(acc[m][t]));
            if (m == 0) READB1(cur ^ 1, un, t);
            if (m > 0 && t == 0) LOADA1(h, kk, m - 1, rs, so);
            SCHED();
          }
        }
        LOADA1(h, kk, 3, rs, so);
        SCHED();
      }
      ctl[4 + w] = i0 + 4;
      SCHED();
    }
  }
  failed |= __builtin_amdgcn_readfirstlane(ctl[3]);
  LSTAMP(9); LSTAMPC(13);
  {
    const int T = (xcd & 3) * 32 + idx;
    const __amdgpu_buffer_rsrc_t rsS = __builtin_amdgcn_make_buffer_rsrc((void*)(wsl + WS_SLAB), 0, (int)SLAB_BYTES, RSRC_W3);
    {
      const int base = ((T * 2 + (q ^ 1)) * 4 + w) * 8192;
#pragma unroll
      for (int m = 0; m < 2; ++m)
#pragma unroll
        for (int t = 0; t < 4; ++t)
          __builtin_amdgcn_raw_buffer_store_b128(__builtin_bit_cast(v4u, acc[2 + m][t]), rsS, lane * 16 + t * 1024, base + m * 4096, AUX_DEV);
    }
    if (failed && lane == 0) __hip_atomic_store(tmo, call, __ATOMIC_RELAXED, __HIP_MEMORY_SCOPE_AGENT);
    WAIT_V0();
    GAS unsigned* rf = (GAS unsigned*)(wsl + WS_RFLAG);
    if (lane == 0) __hip_atomic_store(rf + ((T * 2 + (q ^ 1)) * 4 + w) * 16, call, __ATOMIC_RELAXED, __HIP_MEMORY_SCOPE_AGENT);
    {
      GAS unsigned* fl = rf + ((T * 2 + q) * 4 + w) * 16;
      for (;;) {
        const unsigned fv = __hip_atomic_load(fl, __ATOMIC_RELAXED, __HIP_MEMORY_SCOPE_AGENT);
        const unsigned tv = __hip_atomic_load(tmo, __ATOMIC_RELAXED, __HIP_MEMORY_SCOPE_AGENT);
        if (tv == call) { failed = 1; break; }
        if (fv >= call) break;
        if ((unsigned)__builtin_amdgcn_s_memrealtime() - t0 > DEADLINE_LDS) {
          failed = 1;
          if (lane == 0) __hip_atomic_store(tmo, call, __ATOMIC_RELAXED, __HIP_MEMORY_SCOPE_AGENT);
          break;
        }
        __builtin_amdgcn_s_sleep(1);
      }
    }
    LSTAMP(10);
    __builtin_amdgcn_fence(__ATOMIC_ACQUIRE, "wavefront");
    v4u pr[2][4];
    const int base = ((T * 2 + q) * 4 + w) * 8192;
#pragma unroll
    for (int m = 0; m < 2; ++m)
#pragma unroll
      for (int t = 0; t < 4; ++t) pr[m][t] = __builtin_amdgcn_raw_buffer_load_b128(rsS, lane * 16 + t * 1024, base + m * 4096, AUX_DEV);
#pragma unroll
    for (int m = 0; m < 2; ++m)
#pragma unroll
      for (int t = 0; t < 4; ++t) acc[m][t] += __builtin_bit_cast(f32x4, pr[m][t]);
  }
  const float poison = failed ? __builtin_nanf("") : 0.0f;
#pragma unroll
  for (int m = 0; m < 2; ++m) {
    const int m16 = mt * 16 + w * 4 + 2 * q + m;
#pragma unroll
    for (int jj = 0; jj < 4; ++jj) {
      f32x4 o;
#pragma unroll
      for (int t = 0; t < 4; ++t) {
        const float y = acc[m][t][jj] + poison;
        const float c = fminf(fmaxf(y, -60.f), 60.f);
        o[t] = y / (1.0f + __expf(-c));
      }
      const int row = m16 * 16 + (lane >> 4) * 4 + jj;
      *(f32x4*)(p.out + (size_t)row * ND + n0 + (lane & 15) * 4) = o;
    }
  }
  if (lane == 0) __hip_atomic_store(epoch, call, __ATOMIC_RELAXED, __HIP_MEMORY_SCOPE_AGENT);
}

}

extern "C" void kernel_launch(int rank, int nranks, ncclComm_t comm,
                              hipStream_t stream,
                              void* const* d_in, const int* in_sizes, int n_in,
                              void* d_out, int out_size,
                              const int* in_dtypes, int out_dtype,
                              void* const* d_out_all,
                              void* const* d_ws_all, size_t ws_bytes) {
  if (nranks != NR || (unsigned)rank >= (unsigned)NR || n_in < 2 || in_sizes[0] != KD * KR || in_sizes[1] != KD * ND ||
      out_size != MR * ND || ws_bytes < WS_USED) {
    fprintf(stderr, "kernel_launch: unexpected problem shape\n");
    return;
  }
  Args a;
  a.x = (const float*)d_in[0];
  a.w = (const float*)d_in[1];
  a.out = (float*)d_out;
  for (int p = 0; p < NR; ++p) a.ws[p] = (unsigned char*)d_ws_all[p];
  a.rank = rank;
  a2a_gemm_silu<<<NWG, NWAVES * 64, 0, stream>>>(a);
  HIP_CHECK(hipGetLastError());
}
```

```cpp
#include <hip/hip_runtime.h>
#include <rccl/rccl.h>
#include <cstdio>

#define HIP_CHECK(expr) do { hipError_t _e = (expr); if (_e != hipSuccess) { \
    fprintf(stderr, "HIP error %s at %s:%d\n", hipGetErrorString(_e), __FILE__, __LINE__); \
    return; } } while (0)

namespace {

typedef unsigned v4u __attribute__((ext_vector_type(4)));
typedef float f32x4 __attribute__((ext_vector_type(4)));
typedef _Float16 h8 __attribute__((ext_vector_type(8)));
typedef _Float16 h2 __attribute__((ext_vector_type(2)));
#define LAS __attribute__((address_space(3)))
#define GAS __attribute__((address_space(1)))

#define UDEPTH 4
#define SEND_T0 100
#define SEND_DT 20
#define POLL_SLEEP 4
#define PRIO 3
#define QUIET_E 0
#define QUIET_AT 0
#define QUIET_T 700
#define POLL_WIN 32
#define POLL_T0 0
#define NOSEND 0
#define ROT_R0 11
#define ROT_SH 0
#define ASM_MFMA 0
#define LD_SLOW 0
#define LPRIO 2
#define NOCOMM 0
#define APF 16
#define AUX_W 0
#define AUX_OUT 16

constexpr int NR = 8;
constexpr int MR = 512;
constexpr int KD = 4096, ND = 4096;
constexpr int KR = 512;
constexpr int NWG = 256;
constexpr int NB = 2;
constexpr int NWAVES = 4 + NB + 2;
constexpr int NKS = 32;
constexpr int NE = 64;
constexpr int UD = UDEPTH;
constexpr int RK = 19;
constexpr int BRES_BYTES = RK * 8192;
constexpr int LDS_BYTES = BRES_BYTES + 1024 + 4096;

constexpr unsigned WS_EPOCH = 0;
constexpr unsigned WS_TMO = 64u << 10;
constexpr unsigned WS_RFLAG = 128u << 10;
constexpr unsigned WS_PROG = 256u << 10;
constexpr unsigned WS_STAGE = 1u << 20;
constexpr unsigned STAGE_BYTES = 4u << 20;
constexpr unsigned WS_SLAB = 16u << 20;
constexpr unsigned SLAB_BYTES = 24u << 20;
constexpr unsigned WS_USED = WS_SLAB + SLAB_BYTES;
constexpr unsigned DEADLINE_TICKS = 80000000u;
constexpr unsigned DEADLINE_LDS = 190000000u;

constexpr int AUX_SYS = 17;
constexpr int AUX_DEV = 16;
constexpr int RSRC_W3 = 0x00020000;

#define STAMP(k) do {} while (0)
#define STAMPV(k, v) do {} while (0)
#define LSTAMP(k) do {} while (0)
#define LSTAMPC(k) do {} while (0)

struct Args {
  const float* x;
  const float* w;
  float* out;
  unsigned char* ws[NR];
  int rank;
};

__device__ __forceinline__ unsigned cvt2(float a, float b) {
  h2 r; r[0] = (_Float16)a; r[1] = (_Float16)b; return __builtin_bit_cast(unsigned, r);
}
__device__ __forceinline__ f32x4 mfma(v4u a, v4u b, f32x4 c) {
  return __builtin_amdgcn_mfma_f32_16x16x32_f16(__builtin_bit_cast(h8, a), __builtin_bit_cast(h8, b), c, 0, 0, 0);
}
#define SCHED() __builtin_amdgcn_sched_barrier(0)
#define WAIT_L0() asm volatile("s_waitcnt lgkmcnt(0)" ::: "memory")
#define WAIT_V0() asm volatile("s_waitcnt vmcnt(0)" ::: "memory")

#define STORE128(v, rs, voff, soff, aux) do { __builtin_amdgcn_raw_buffer_store_b128(v, rs, voff, soff, aux);        \
    SCHED(); asm volatile("s_nop 1"); SCHED(); } while (0)

__device__ __forceinline__ int rot_fwd(int pos, int mt) {
  return (mt == 0 || pos < ROT_R0) ? pos : ROT_R0 + (pos - ROT_R0 + ROT_SH) % (32 - ROT_R0);
}
__device__ __forceinline__ int kseq(int i, int q, int rank, int mt) {
  if (mt != 0 && i >= ROT_R0) i = ROT_R0 + (i - ROT_R0 + (32 - ROT_R0) - ROT_SH) % (32 - ROT_R0);
  int d, ff;
  if (i < 4) { d = 0; ff = i; }
  else { int j = i - 4; ff = j / 7; d = 1 + j % 7; }
  return ((((rank + d) & 7) << 4) + 4 * ff + q) & 127;
}

__global__ void __launch_bounds__(NWAVES * 64, 2) a2a_gemm_silu(Args p) {
  __shared__ __attribute__((aligned(16))) unsigned char smem[LDS_BYTES];
  LAS unsigned char* L = (LAS unsigned char*)smem;
  volatile LAS unsigned* ctl = (volatile LAS unsigned*)(L + BRES_BYTES);
  const int tid = threadIdx.x, lane = tid & 63, w = __builtin_amdgcn_readfirstlane(tid >> 6);
  const int b = blockIdx.x, xcd = b & 7, idx = b >> 3;
  unsigned char* const wsl = p.ws[p.rank];
  const unsigned t0 = (unsigned)__builtin_amdgcn_s_memrealtime();
  GAS unsigned* const epoch = (GAS unsigned*)(wsl + WS_EPOCH) + b * 32 + w;

  if (w == 4 + NB) {
    __builtin_amdgcn_s_setprio(PRIO);
    const int dd = b >> 5, u = (b & 24) | ((b + dd) & 7);
    const int dest = (p.rank + dd) & 7, f = u >> 1, mts = u & 1, qc = f & 3, ff = f >> 2;
    const unsigned ev = __hip_atomic_load(epoch, __ATOMIC_RELAXED, __HIP_MEMORY_SCOPE_AGENT);
    const __amdgpu_buffer_rsrc_t rsX = __builtin_amdgcn_make_buffer_rsrc((void*)p.x, 0, KD * KR * 4, RSRC_W3);
    const int voffX = (lane & 15) * (KR * 4) + (lane >> 4) * 32;
    const int soX = (dest * MR + mts * 256) * (KR * 4) + f * 128;
    f32x4 xr[2][8][2];
#pragma unroll
    for (int h = 0; h < 2; ++h)
#pragma unroll
      for (int m = 0; m < 8; ++m) {
        const int so = soX + ((m >> 1) * 4 + ((h ^ (qc >> 1) ^ 1) * 2) + (m & 1)) * (16 * KR * 4);
        xr[h][m][0] = __builtin_bit_cast(f32x4, __builtin_amdgcn_raw_buffer_load_b128(rsX, voffX, so, 0));
        xr[h][m][1] = __builtin_bit_cast(f32x4, __builtin_amdgcn_raw_buffer_load_b128(rsX, voffX + 16, so, 0));
      }
    SCHED();
    __builtin_amdgcn_s_barrier();
    SCHED();
    const unsigned call = __builtin_amdgcn_readfirstlane(ev) + 1u;
    const __amdgpu_buffer_rsrc_t rsD = __builtin_amdgcn_make_buffer_rsrc((void*)(p.ws[dest] + WS_STAGE), 0, 2 * STAGE_BYTES, RSRC_W3);
    const int d = (8 - dd) & 7;
    const int pos = rot_fwd(d == 0 ? ff : 4 + ff * 7 + (d - 1), mts);
    int ord = 0;
#pragma unroll
    for (int o = 0; o < 4; ++o) ord += rot_fwd(4 + o * 7 + (d - 1), mts) < pos ? 1 : 0;
    const int us = ord * 8 + (u & 7);
    GAS unsigned* const fl = (GAS unsigned*)(p.ws[dest] + WS_PROG) + (qc * 2 + mts) * NE * 4;
    const bool both = dd == 0 || ord < 2;
#pragma unroll
    for (int h = 0; h < 2; ++h) {
      const unsigned tl = dd == 0 ? 0u : SEND_T0 + (ord < 2 ? 2 * us : 16 + h * 16 + us) * SEND_DT;
      while ((unsigned)__builtin_amdgcn_s_memrealtime() - t0 < tl) __builtin_amdgcn_s_sleep(2);
      SCHED();
      STAMP(1 + 4 * h);
      const int soD = (int)((call & 1u) * STAGE_BYTES) + ((p.rank * 16 + f) * 32 + mts * 16 + h * 8) * 1024;
#pragma unroll
      for (int m = 0; m < 8; ++m) {
        v4u o;
        o[0] = cvt2(xr[h][m][0][0], xr[h][m][0][1]); o[1] = cvt2(xr[h][m][0][2], xr[h][m][0][3]);
        o[2] = cvt2(xr[h][m][1][0], xr[h][m][1][1]); o[3] = cvt2(xr[h][m][1][2], xr[h][m][1][3]);
        if (!(NOSEND && call > 4u && dd != 0)) STORE128(o, rsD, lane * 16 + (m & 3) * 1024, soD + (m >> 2) * 4096, AUX_SYS);
      }
      STAMP(2 + 4 * h);
      if (h == 1 || !both) {
        WAIT_V0();
        STAMP(3 + 4 * h);
        const int hh = h - lane;
        if (lane < (h == 1 && both ? 2 : 1))
          __hip_atomic_store(fl + (pos < 16 ? 2 * pos + hh : 16 + hh * 16 + pos) * 4, call, __ATOMIC_RELAXED, __HIP_MEMORY_SCOPE_SYSTEM);
      }
    }
    if (lane == 0) __hip_atomic_store(epoch, call, __ATOMIC_RELAXED, __HIP_MEMORY_SCOPE_AGENT);
    STAMPV(0, t0); STAMPV(9, u);
    return;
  }

  const int q = xcd >> 1, mt = idx & 1, ntl = idx >> 1;
  const int n0 = (xcd & 1) * 2048 + ntl * 128;
  const int tile = (xcd & 1) * 32 + idx;
  const int ktab = kseq(lane & 31, q, p.rank, mt);
#define KS(i) (__builtin_amdgcn_readlane(ktab, (i) & 31))
  const unsigned ctl_addr = BRES_BYTES;
  if (tid < 16) ctl[tid] = 0;
  WAIT_L0();
  __builtin_amdgcn_s_barrier();

  if (w == 5 + NB) {
    __builtin_amdgcn_s_setprio(PRIO);
    const unsigned call = __builtin_amdgcn_readfirstlane(__hip_atomic_load(epoch, __ATOMIC_RELAXED, __HIP_MEMORY_SCOPE_AGENT)) + 1u;
    GAS unsigned* const tmo = (GAS unsigned*)(wsl + WS_TMO);
    GAS unsigned* const pa = (GAS unsigned*)(wsl + WS_PROG) + (q * 2 + mt) * NE * 4;
    GAS unsigned* const pr = (GAS unsigned*)(wsl + WS_RFLAG) + (tile * 4 + q) * 32 + (lane < 12 ? lane : 0);
    const __amdgpu_buffer_rsrc_t rsA = __builtin_amdgcn_make_buffer_rsrc((void*)(wsl + WS_STAGE + (call & 1u) * STAGE_BYTES), 0, (int)STAGE_BYTES, RSRC_W3);
    int pub = 0, nextpf = ntl + APF;
    while ((unsigned)__builtin_amdgcn_s_memrealtime() - t0 < POLL_T0) __builtin_amdgcn_s_sleep(4);
    unsigned seen = 0;
    for (unsigned it = 1;; ++it) {
      if (pub < NE) {
        const int s = min((pub & ~7) + (lane & (POLL_WIN - 1)), NE - 1);
        const unsigned v = __hip_atomic_load(pa + s * 4, __ATOMIC_RELAXED, __HIP_MEMORY_SCOPE_SYSTEM);
        const unsigned missing = (NOCOMM && call > 4u) ? ((unsigned)__builtin_amdgcn_s_memrealtime() - t0 < NOCOMM ? 1u : 0u) : (unsigned)__builtin_amdgcn_ballot_w64((int)(v - call) < 0);
        const int rdy = min((pub & ~7) + (missing ? __builtin_ctz(missing) : POLL_WIN), NE);
        if (rdy != pub) { pub = rdy; ctl[2] = rdy; }
        while (nextpf < pub) {
          const int so = (nextpf < 32 ? KS(nextpf >> 1) * 32 + (nextpf & 1) * 8 : KS(16 + (nextpf & 15)) * 32 + ((nextpf >> 4) & 1) * 8) * 1024 + mt * 16384;
          v4u t[8];
#pragma unroll
          for (int i = 0; i < 8; ++i) t[i] = __builtin_amdgcn_raw_buffer_load_b128(rsA, lane * 16 + (i & 3) * 1024, so + (i >> 2) * 4096, AUX_SYS);
          asm volatile("" :: "v"(t[0]), "v"(t[1]), "v"(t[2]), "v"(t[3]), "v"(t[4]), "v"(t[5]), "v"(t[6]), "v"(t[7]));
          nextpf += 16;
        }
        if (pub >= NE) STAMP(0);
      } else {
        const unsigned v = __hip_atomic_load(pr, __ATOMIC_RELAXED, __HIP_MEMORY_SCOPE_AGENT);
        const unsigned got = (unsigned)__builtin_amdgcn_ballot_w64((int)(v - call) >= 0) & 0xfffu;
        if (got != seen) { seen = got; ctl[3] = got; }
        if (seen == 0xfffu) break;
      }
      if ((it & 63u) == 0u) {
        if ((unsigned)__builtin_amdgcn_s_memrealtime() - t0 > DEADLINE_TICKS ||
            __hip_atomic_load(tmo, __ATOMIC_RELAXED, __HIP_MEMORY_SCOPE_AGENT) == call) {
          if (lane == 0) __hip_atomic_store(tmo, call, __ATOMIC_RELAXED, __HIP_MEMORY_SCOPE_AGENT);
          ctl[8] = 1;
          WAIT_L0();
          ctl[2] = NE;
          ctl[3] = 0xfffu;
          break;
        }
      }
      __builtin_amdgcn_s_sleep(POLL_SLEEP);
    }
    STAMP(1);
    if (lane == 0) __hip_atomic_store(epoch, call, __ATOMIC_RELAXED, __HIP_MEMORY_SCOPE_AGENT);
    return;
  }
  if (w >= 4) {
    __builtin_amdgcn_s_setprio(LPRIO);
    const int j = w - 4;
    const int voffB = (lane >> 4) * 8 * (ND * 4) + (lane & 15) * 16 + (n0 + (j ^ (q & 1) ^ 1) * 64) * 4;
    f32x4 br[UD][8];
#define LOADB(r, u) do { const int u_ = (u); const int so_ = KS(u_) * (32 * ND * 4);                                 \
      const __amdgpu_buffer_rsrc_t rs_ = __builtin_amdgcn_make_buffer_rsrc((void*)p.w, 0, u_ < NKS ? KD * ND * 4 : 0, RSRC_W3); \
      _Pragma("unroll") for (int jj = 0; jj < 8; ++jj)                                                              \
        br[r][jj] = __builtin_bit_cast(f32x4, __builtin_amdgcn_raw_buffer_load_b128(rs_, voffB, so_ + jj * (ND * 4), AUX_W)); } while (0)
#pragma unroll
    for (int r = 0; r < UD; ++r) {
      LOADB(r, r); SCHED();
    }
    int freed = RK;
#pragma unroll 1
    for (int n_ = 0; n_ < NKS; n_ += UD) {
      LSTAMP(n_ / UD);
#pragma unroll
      for (int r = 0; r < UD; ++r) {
        const int u = n_ + r;
        v4u bf[4];
#pragma unroll
        for (int t = 0; t < 4; ++t)
#pragma unroll
          for (int jj = 0; jj < 4; ++jj) bf[t][jj] = cvt2(br[r][2 * jj][t], br[r][2 * jj + 1][t]);
        asm volatile("" : "+v"(bf[0]), "+v"(bf[1]), "+v"(bf[2]), "+v"(bf[3]));
        SCHED();
        LOADB(r, u + UD);
        SCHED();
        {
          unsigned t0_, t1_, t2_, t3_;
          asm volatile("s_cmp_lt_i32 %[uu], %[fr]\n\ts_cbranch_scc1 .Lfok%=\n"
                       ".Lfp%=:\n\tds_read_b32 %[t0], %[ad] offset:16\n\tds_read_b32 %[t1], %[ad] offset:20\n\t"
                       "ds_read_b32 %[t2], %[ad] offset:24\n\tds_read_b32 %[t3], %[ad] offset:28\n\t"
                       "s_memrealtime vcc\n\ts_waitcnt lgkmcnt(0)\n\t"
                       "v_min_u32 %[t0], %[t0], %[t1]\n\tv_min3_u32 %[t0], %[t0], %[t2], %[t3]\n\ts_nop 0\n\t"
                       "v_readfirstlane_b32 %[fr], %[t0]\n\ts_add_i32 %[fr], %[fr], %[ru]\n\t"
                       "s_cmp_lt_i32 %[uu], %[fr]\n\ts_cbranch_scc1 .Lfok%=\n\t"
                       "s_sleep 4\n\ts_sub_u32 vcc_lo, vcc_lo, %[ts]\n\ts_cmp_lt_u32 vcc_lo, %[dl]\n\ts_cbranch_scc1 .Lfp%=\n"
                       ".Lfok%=:"
                       : [fr] "+&s"(freed), [t0] "=&v"(t0_), [t1] "=&v"(t1_), [t2] "=&v"(t2_), [t3] "=&v"(t3_)
                       : [uu] "s"(u < NKS ? u : 0), [ad] "v"(ctl_addr), [ru] "i"(RK), [ts] "s"(t0), [dl] "s"(DEADLINE_LDS)
                       : "scc", "vcc", "memory");
          freed = __builtin_amdgcn_readfirstlane(freed);
        }
        LAS v4u* slot = (LAS v4u*)(L + (u < NKS ? (u < RK ? u : u - RK) * 8192 + j * 4096 : BRES_BYTES + 1024));
#pragma unroll
        for (int t = 0; t < 4; ++t) slot[t * 64 + lane] = bf[t];
        WAIT_L0();
        ctl[j] = u + 1;
        SCHED();
      }
    }
    STAMP(0);
    return;
  }

  const unsigned call = __builtin_amdgcn_readfirstlane(__hip_atomic_load(epoch, __ATOMIC_RELAXED, __HIP_MEMORY_SCOPE_AGENT)) + 1u;
  const __amdgpu_buffer_rsrc_t rsA = __builtin_amdgcn_make_buffer_rsrc((void*)(wsl + WS_STAGE + (call & 1u) * STAGE_BYTES), 0, (int)STAGE_BYTES, RSRC_W3);
  const int voffA = lane * 16 + mt * 16384 + w * 2048;
  f32x4 acc[4][8];
#pragma unroll
  for (int m = 0; m < 4; ++m)
#pragma unroll
    for (int t = 0; t < 8; ++t) acc[m][t] = f32x4{0.f, 0.f, 0.f, 0.f};
  v4u a[16][2];
  const int atab = ktab << 15;
#define SOK(i) __builtin_amdgcn_readlane(atab, i)
#define LOADA1(r, m, so) a[r][m] = __builtin_amdgcn_raw_buffer_load_b128(rsA, voffA + (m) * 1024, so, AUX_SYS)
  int readyb = 0, readya = 0;
  unsigned failed = 0;
#define WAITAB(ub, ua) do { unsigned t0_, t1_, t2_;                                                                  \
    asm volatile("s_cmp_lt_i32 %[u1], %[rb]\n\ts_cselect_b32 vcc_lo, %[ra], 0\n\t"                                   \
                 "s_cmp_lt_i32 %[u2], vcc_lo\n\ts_cbranch_scc1 .Lbok%=\n"                                            \
                 ".Lbp%=:\n\tds_read_b32 %[t0], %[ad]\n\tds_read_b32 %[t1], %[ad] offset:4\n\tds_read_b32 %[t2], %[ad] offset:8\n\t" \
                 "s_memrealtime vcc\n\ts_waitcnt lgkmcnt(0)\n\t"                                                     \
                 "v_min_u32 %[t0], %[t0], %[t1]\n\tv_readfirstlane_b32 %[ra], %[t2]\n\tv_readfirstlane_b32 %[rb], %[t0]\n\t" \
                 "s_sub_u32 vcc_hi, vcc_lo, %[ts]\n\t"                                                               \
                 "s_cmp_lt_i32 %[u1], %[rb]\n\ts_cselect_b32 vcc_lo, %[ra], 0\n\t"                                   \
                 "s_cmp_lt_i32 %[u2], vcc_lo\n\ts_cbranch_scc1 .Lbok%=\n\t"                                          \
                 "s_sleep 2\n\ts_cmp_lt_u32 vcc_hi, %[dl]\n\ts_cbranch_scc1 .Lbp%=\n\t"                              \
                 "s_mov_b32 %[fl], 1\n"                                                                              \
                 ".Lbok%=:"                                                                                          \
                 : [rb] "+&s"(readyb), [ra] "+&s"(readya), [fl] "+&s"(failed), [t0] "=&v"(t0_), [t1] "=&v"(t1_), [t2] "=&v"(t2_) \
                 : [u1] "s"(ub), [u2] "s"(ua), [ad] "v"(ctl_addr), [ts] "s"(t0), [dl] "s"(DEADLINE_LDS) : "scc", "vcc", "memory"); \
    readyb = __builtin_amdgcn_readfirstlane(readyb); readya = __builtin_amdgcn_readfirstlane(readya);                \
    failed = __builtin_amdgcn_readfirstlane(failed); } while (0)
#define WAITP(bit) do { unsigned t0_;                                                                                \
    asm volatile(".Lpp%=:\n\tds_read_b32 %[t0], %[ad] offset:12\n\ts_memrealtime vcc\n\ts_waitcnt lgkmcnt(0)\n\t"    \
                 "v_readfirstlane_b32 vcc_hi, %[t0]\n\ts_sub_u32 vcc_lo, vcc_lo, %[ts]\n\t"                          \
                 "s_and_b32 vcc_hi, vcc_hi, %[bm]\n\ts_cbranch_scc1 .Lpok%=\n\t"                                     \
                 "s_sleep 1\n\ts_cmp_lt_u32 vcc_lo, %[dl]\n\ts_cbranch_scc1 .Lpp%=\n\t"                              \
                 "s_mov_b32 %[fl], 1\n"                                                                              \
                 ".Lpok%=:"                                                                                          \
                 : [fl] "+&s"(failed), [t0] "=&v"(t0_)                                                               \
                 : [bm] "s"(bit), [ad] "v"(ctl_addr), [ts] "s"(t0), [dl] "s"(DEADLINE_LDS) : "scc", "vcc", "memory"); \
    failed = __builtin_amdgcn_readfirstlane(failed); } while (0)
  v4u bq[2][4];
  const LAS v4u* lbase = (const LAS v4u*)L + lane;
#define SLOT(i) ((i) < RK ? (i) : (i) - RK)
#define READB1(buf, slot, g, t) bq[buf][t] = lbase[(((slot) * 2 + (g)) * 4 + (t)) * 64]
  __builtin_amdgcn_s_waitcnt(0xC07F);
#pragma unroll
  for (int r2 = 0; r2 < 8; r2 += 2) {
    WAITAB(-1, r2 + 1);
    if (r2 == 0) { LSTAMP(0); LSTAMPC(12); }
    const int so = SOK(r2 >> 1);
    LOADA1(r2, 0, so); LOADA1(r2, 1, so); LOADA1(r2 + 1, 0, so + 8192); LOADA1(r2 + 1, 1, so + 8192);
    SCHED();
  }
  WAITAB(0, 0);
#pragma unroll
  for (int t = 0; t < 4; ++t) READB1(0, 0, 0, t);
#define QUAD(r, m0, g, bc, RD, sl, gn, RF, REFILL) do {                                                              \
    if (RD) { __builtin_amdgcn_s_waitcnt(0xC07F); SCHED(); }                     \
    _Pragma("unroll") for (int m = 0; m < 2; ++m) {                                                                  \
      _Pragma("unroll") for (int t = 0; t < 4; ++t) {                                                                \
        acc[(m0) + m][(g) * 4 + t] = mfma(a[r][m], bq[bc][t], acc[(m0) + m][(g) * 4 + t]);                           \
        if (!ASM_MFMA) asm volatile("" : "+v"(acc[(m0) + m][(g) * 4 + t]));                 \
        if (m == 0 && (RD)) READB1((bc) ^ 1, sl, gn, t);                                                             \
        if (m == 1 && t == 0 && (RF)) { REFILL(r, 0); }                                                              \
        SCHED();                                                                                                     \
      }                                                                                                              \
    }                                                                                                                \
    if (RF) { REFILL(r, 1); } SCHED(); } while (0)
#pragma unroll 1
  for (int it = 0; it < 4; ++it) {
    const int k0 = it * 4;
    const bool last = it == 3;
    const int odd = last ? 0 : 8192;
    LSTAMP(1 + it);
#pragma unroll
    for (int kk = 0; kk < 4; ++kk) {
      if ((kk & 1) == 0) { WAITAB(k0 + kk + 2, 2 * (k0 + kk) + 11); SCHED(); }
      const int so0 = SOK(last ? 16 + 2 * kk : k0 + 4 + kk), so1 = SOK(last ? 17 + 2 * kk : k0 + 4 + kk) + odd;
#define RF_0(r, m) LOADA1(r, m, so0)
#define RF_1(r, m) LOADA1(r, m, so1)
      QUAD(2 * kk, 0, 0, 0, true, k0 + kk, 1, false, RF_0);
      QUAD(2 * kk + 1, 2, 0, 0, false, 0, 0, false, RF_1);
      QUAD(2 * kk, 0, 1, 1, true, k0 + kk + 1, 0, true, RF_0);
      QUAD(2 * kk + 1, 2, 1, 1, false, 0, 0, true, RF_1);
      if (kk & 1) { ctl[4 + w] = k0 + kk + 1; SCHED(); }
    }
  }
  const __amdgpu_buffer_rsrc_t rsS = __builtin_amdgcn_make_buffer_rsrc((void*)(wsl + WS_SLAB), 0, (int)SLAB_BYTES, RSRC_W3);
  GAS unsigned* const rf = (GAS unsigned*)(wsl + WS_RFLAG);
#define RF_S(r, m) LOADA1(r, m, so)
#pragma unroll 1
  for (int it = 0; it < 2; ++it) {
    const int k0 = 16 + it * 8;
    const int kn = it ? 16 : 24, pr = it ? 8192 : 0;
    LSTAMP(5 + it);
    if (it == 0) LSTAMPC(14);
#pragma unroll
    for (int r = 0; r < 8; ++r) {
      if ((r & 3) == 0) { WAITAB(min(k0 + r + 4, NKS - 1), 32 + it * 8 + r + 11); SCHED(); }
      const int so = SOK(kn + r) + pr;
      const int sl = SLOT(k0 + r), nx = k0 + r + 1, sn = nx == NKS ? 16 : SLOT(nx);
      QUAD(r, 0, 0, 0, true, sl, 1, false, RF_S);
      QUAD(r, 0, 1, 1, true, sn, 0, true, RF_S);
    }
  }
#define PUT(j) do { if (ASM_MFMA) { asm volatile("s_nop 7\n\ts_nop 7"); SCHED(); } const int base_ = (((tile * 4 + (q ^ (3 - (j)))) * 3 + (j)) * 4 + w) * 8192;                        \
    _Pragma("unroll") for (int m = 0; m < 2; ++m) _Pragma("unroll") for (int t = 0; t < 4; ++t)                      \
      STORE128(__builtin_bit_cast(v4u, acc[((j) >> 1) * 2 + m][((j) & 1) * 4 + t]), rsS, lane * 16 + t * 1024, base_ + m * 4096, AUX_DEV); } while (0)
  PUT(0); PUT(1);
  LSTAMP(7);
  WAITAB(-1, NE - 1);
#pragma unroll
  for (int r = 8; r < 16; ++r) { const int so = SOK(16 + r) + 8192; LOADA1(r, 0, so); LOADA1(r, 1, so); SCHED(); }
  v4u bx[3][4];
#define READX(buf, k, g, t) bx[buf][t] = lbase[((SLOT(k) * 2 + (g)) * 4 + (t)) * 64]
#pragma unroll
  for (int t = 0; t < 4; ++t) { bx[0][t] = bq[0][t]; READX(1, 17, 0, t); }
  const int pbase = ((tile * 4 + q) * 3 * 4 + w) * 8192;
  v4u pp[3][8];
#define GETP(j, i) pp[j][i] = __builtin_amdgcn_raw_buffer_load_b128(rsS, lane * 16 + ((i) & 3) * 1024, pbase + (j) * 32768 + ((i) >> 2) * 4096, AUX_DEV)
#pragma unroll
  for (int s = 0; s < 32; ++s) {
    const int g = s >> 4, r = s & 15, sn = s + 2;
    if (s == 6) {
      asm volatile("s_waitcnt vmcnt(16)\n\tglobal_store_dword %[z], %[v], %[p0] sc1\n\tglobal_store_dword %[z], %[v], %[p1] sc1"
                   :: [z] "v"(0), [v] "v"(call), [p0] "s"(rf + (tile * 4 + (q ^ 3)) * 32 + w),
                      [p1] "s"(rf + (tile * 4 + (q ^ 2)) * 32 + 4 + w) : "memory");
      SCHED();
    }
    if (s == 16) { PUT(2); LSTAMP(8); }
    if (s == 23) {
      asm volatile("s_waitcnt vmcnt(0)\n\tglobal_store_dword %[z], %[v], %[p0] sc1"
                   :: [z] "v"(0), [v] "v"(call), [p0] "s"(rf + (tile * 4 + (q ^ 1)) * 32 + 8 + w) : "memory");
      SCHED();
    }
    if (s == 24 || s == 28) { WAITP(1u << (s - 24 + w)); SCHED(); }
    if (s < 31) { __builtin_amdgcn_s_waitcnt(0xC47F); SCHED(); }
#pragma unroll
    for (int m = 0; m < 2; ++m)
#pragma unroll
      for (int t = 0; t < 4; ++t) {
        acc[2 + m][g * 4 + t] = mfma(a[r][m], bx[s % 3][t], acc[2 + m][g * 4 + t]);
        if (!ASM_MFMA) asm volatile("" : "+v"(acc[2 + m][g * 4 + t]));
        if (m == 0 && sn < 32) READX(sn % 3, 16 + (sn & 15), sn >> 4, t);
        if (m == 1 && s >= 24 && (t & 1) == 0) GETP((s - 24) >> 2, (s & 3) * 2 + (t >> 1));
        SCHED();
      }
  }
  if (ASM_MFMA) { asm volatile("s_nop 7\n\ts_nop 7"); SCHED(); }
  LSTAMP(9); LSTAMPC(13);
  WAITP(1u << (8 + w));
  LSTAMP(10);
#pragma unroll
  for (int i = 0; i < 8; ++i) GETP(2, i);
  SCHED();
  failed |= __builtin_amdgcn_readfirstlane(ctl[8]);
#pragma unroll
  for (int j = 0; j < 3; ++j)
#pragma unroll
    for (int i = 0; i < 8; ++i) acc[2 + (i >> 2)][4 + (i & 3)] += __builtin_bit_cast(f32x4, pp[j][i]);
  const float poison = failed ? __builtin_nanf("") : 0.0f;
  const __amdgpu_buffer_rsrc_t rsO = __builtin_amdgcn_make_buffer_rsrc((void*)p.out, 0, MR * ND * 4, RSRC_W3);
  const int voffO = ((lane >> 4) * 4) * (ND * 4) + (lane & 15) * 16;
#pragma unroll
  for (int m = 0; m < 2; ++m)
#pragma unroll
    for (int jj = 0; jj < 4; ++jj) {
      f32x4 o;
#pragma unroll
      for (int t = 0; t < 4; ++t) {
        const float y = acc[2 + m][4 + t][jj] + poison;
        const float c = fminf(fmaxf(y, -60.f), 60.f);
        o[t] = y * __builtin_amdgcn_rcpf(1.0f + __expf(-c));
      }
      STORE128(__builtin_bit_cast(v4u, o), rsO, voffO, ((mt * 256 + w * 64 + (q >> 1) * 32 + m * 16 + jj) * ND + n0 + (q & 1) * 64) * 4, AUX_OUT);
    }
  if (lane == 0) __hip_atomic_store(epoch, call, __ATOMIC_RELAXED, __HIP_MEMORY_SCOPE_AGENT);
}

}

extern "C" void kernel_launch(int rank, int nranks, ncclComm_t comm,
                              hipStream_t stream,
                              void* const* d_in, const int* in_sizes, int n_in,
                              void* d_out, int out_size,
                              const int* in_dtypes, int out_dtype,
                              void* const* d_out_all,
                              void* const* d_ws_all, size_t ws_bytes) {
  if (nranks != NR || (unsigned)rank >= (unsigned)NR || n_in < 2 || in_sizes[0] != KD * KR || in_sizes[1] != KD * ND ||
      out_size != MR * ND || ws_bytes < WS_USED) {
    fprintf(stderr, "kernel_launch: unexpected problem shape\n");
    return;
  }
  Args a;
  a.x = (const float*)d_in[0];
  a.w = (const float*)d_in[1];
  a.out = (float*)d_out;
  for (int p = 0; p < NR; ++p) a.ws[p] = (unsigned char*)d_ws_all[p];
  a.rank = rank;
  a2a_gemm_silu<<<NWG, NWAVES * 64, 0, stream>>>(a);
  HIP_CHECK(hipGetLastError());
}
```
